```python
import numpy as np

kernel_name = "gpu_ag_gemm_m4096_k4096_n16384_bf16_silu"


ABSMAX_RTOL = 0.008

COMPUTE_DTYPE = "bfloat16"
EPILOGUE = "silu"
IN_QUANT = None
OUT_QUANT = None
IN_DTYPES = {"x": "bfloat16", "w_mat": "bfloat16"}
WORLD_SIZE = 8

def _to_bf16(x):
    u = np.ascontiguousarray(x, dtype=np.float32).view(np.uint32)
    return ((u + 0x7FFF + ((u >> 16) & 1)) & 0xFFFF0000).view(np.float32)

def setup_inputs():

    x = _to_bf16(np.random.randn(4096, 4096).astype(np.float32))
    w_mat = _to_bf16((np.random.randn(4096, 16384) / np.sqrt(4096)).astype(np.float32))
    return {"x": x, "w_mat": w_mat}

def _epilogue(y):
    return (y / (1.0 + np.exp(-np.clip(y, -60.0, 60.0)))).astype(np.float32)

def reference(x, w_mat):
    y = (x @ w_mat).astype(np.float32)
    return _epilogue(y)
```

```cpp
#include <hip/hip_runtime.h>
#include <hip/hip_bf16.h>
#include <rccl/rccl.h>
#include <cstdio>
#include <type_traits>

#define HIP_CHECK(expr) do { hipError_t _e = (expr); if (_e != hipSuccess) { \
    fprintf(stderr, "HIP error %s at %s:%d\n", hipGetErrorString(_e), __FILE__, __LINE__); \
    return; } } while (0)

namespace {

typedef short bf16x8 __attribute__((ext_vector_type(8)));
typedef short bf16x4 __attribute__((ext_vector_type(4)));
typedef float f32x4 __attribute__((ext_vector_type(4)));
typedef unsigned u32x2 __attribute__((ext_vector_type(2)));
typedef unsigned u32x4 __attribute__((ext_vector_type(4)));

constexpr int NR = 8;
constexpr int M_ALL = 4096, K_ALL = 4096, N_LOC = 2048, M_LOC = 512;
constexpr int BM = 128, BN = 256, BK = 32;
constexpr int NSLICE = K_ALL / BK;
constexpr int WSL = BK * BN * 2;
constexpr int XUN = BM * 64 * 2;
constexpr int LDS_W0 = 0, LDS_X0 = 3 * WSL;
constexpr int LDS_BYTES = 3 * WSL + 2 * XUN;
constexpr int KC = 256, NCH = K_ALL / KC;
constexpr int SPC = KC / BK;
constexpr size_t CHUNK_B = (size_t)M_LOC * KC * 2;
constexpr size_t SRC_B = CHUNK_B * NCH;

constexpr int WPP = 4;
constexpr int NPUSH = (NR - 1) * WPP;
constexpr int NBAND = WPP * 4;
constexpr int BAND_ROWS = M_LOC / NBAND;
constexpr int NGEMM = 256;
constexpr int GRID = NPUSH + NGEMM;

constexpr size_t WS_STAGE = 0;
constexpr size_t WS_FLAG = 2 * NR * SRC_B;
constexpr size_t WS_CTR = WS_FLAG + (size_t)NR * NCH * 128;
constexpr size_t WS_NEED = WS_CTR + 1024;
constexpr unsigned long long DEADLINE_TICKS = 50000000ull;

struct Params {
  const unsigned short* x;
  const unsigned short* w;
  unsigned short* out;
  char* ws;
  void* const* ws_all;
  int rank;
};

#define WAIT_V(n) asm volatile("s_waitcnt vmcnt(%0)" :: "i"(n) : "memory")
#define WAIT_L(n) asm volatile("s_waitcnt lgkmcnt(%0)" :: "i"(n) : "memory")
#define BAR() asm volatile("s_barrier" ::: "memory")
#define SB() __builtin_amdgcn_sched_barrier(0)

__device__ __forceinline__ float silu_f(float y) {
  const float c = fminf(fmaxf(y, -60.f), 60.f);
  const float e = __expf(-c);
  return y * __builtin_amdgcn_rcpf(1.f + e);
}
__device__ __forceinline__ unsigned pack_bf16(float a, float b) {
  __hip_bfloat162 v = __float22bfloat162_rn(make_float2(a, b));
  return *reinterpret_cast<unsigned*>(&v);
}

#define P_LD(r, vo, sb) asm volatile("global_load_dwordx4 %0, %1, %2" : "=v"(r) : "v"(vo), "s"(sb) : "memory")
#define P_ST(vo, r, sb, imm) asm volatile("global_store_dwordx4 %0, %1, %2 offset:%3 sc0 sc1\n\ts_nop 1" :: "v"(vo), "v"(r), "s"(sb), "i"(imm) : "memory")
#define P_PIN(n, R) asm volatile("s_waitcnt vmcnt(%8)" : "+v"(R[0]), "+v"(R[1]), "+v"(R[2]), "+v"(R[3]), "+v"(R[4]), "+v"(R[5]), "+v"(R[6]), "+v"(R[7]) : "i"(n) : "memory")

__device__ __forceinline__ void pusher(const Params& P, unsigned call, int b, int wv, int l) {
  __builtin_amdgcn_s_setprio(3);
  const int peer = (P.rank + 1 + (b / WPP)) & (NR - 1);
  const int band = (b % WPP) * 4 + wv;
  char* pws;
  {
    const unsigned long long a = (unsigned long long)P.ws_all[peer];
    const unsigned lo = __builtin_amdgcn_readfirstlane((unsigned)a), hi = __builtin_amdgcn_readfirstlane((unsigned)(a >> 32));
    pws = (char*)(((unsigned long long)hi << 32) | lo);
  }
  const char* sb = (const char*)P.x + (size_t)(band * BAND_ROWS) * (K_ALL * 2);
  char* db = pws + WS_STAGE + (size_t)((call & 1) * NR + P.rank) * SRC_B + (size_t)(band * BAND_ROWS) * (KC * 2);
  unsigned* fl = (unsigned*)(pws + WS_FLAG + (size_t)P.rank * NCH * 128) + band;
  unsigned so[8];
  for (int i = 0; i < 8; ++i) so[i] = (unsigned)((l >> 5) + 2 * i) * (K_ALL * 2) + (l & 31) * 16;
  const unsigned do0 = l * 16, do1 = l * 16 + 4096;
  u32x4 A[8], B[8];
  auto ldh = [&](u32x4* R, int c, int h) {
    const char* s = sb + (size_t)c * (KC * 2) + (size_t)h * 16 * (K_ALL * 2);
    for (int i = 0; i < 8; ++i) P_LD(R[i], so[i], s);
  };
  auto sth = [&](u32x4* R, int c, int h) {
    char* d = db + (size_t)c * CHUNK_B + h * 8192;
    for (int i = 0; i < 4; ++i) P_ST(do0, R[i], d, i * 1024);
    for (int i = 0; i < 4; ++i) P_ST(do1, R[4 + i], d, i * 1024);
  };
  auto flag = [&](int c) {
    if (l == 0) __hip_atomic_store(fl + c * 32, call, __ATOMIC_RELAXED, __HIP_MEMORY_SCOPE_SYSTEM);
  };
  ldh(A, 0, 0); ldh(B, 0, 1);
  P_PIN(8, A); sth(A, 0, 0); ldh(A, 1, 0);
  P_PIN(16, B); sth(B, 0, 1); ldh(B, 1, 1);
  for (int c = 1; c < NCH - 1; ++c) {
    P_PIN(16, A); sth(A, c, 0); ldh(A, c + 1, 0);
    P_PIN(16, B);
    flag(c - 1);
    sth(B, c, 1); ldh(B, c + 1, 1);
  }
  P_PIN(16, A); sth(A, NCH - 1, 0);
  P_PIN(8, B);
  flag(NCH - 2);
  sth(B, NCH - 1, 1);
  WAIT_V(0);
  flag(NCH - 1);
}

__device__ __forceinline__ void wait_flags(const unsigned* fl4, unsigned call, unsigned* tmo, int l) {
  const unsigned long long t0 = __builtin_amdgcn_s_memrealtime();
  for (;;) {
    unsigned v = call;
    if (l < 4) v = __hip_atomic_load(fl4 + l, __ATOMIC_RELAXED, __HIP_MEMORY_SCOPE_SYSTEM);
    if (__builtin_amdgcn_ballot_w64(v < call) == 0) break;
    if (__hip_atomic_load(tmo, __ATOMIC_RELAXED, __HIP_MEMORY_SCOPE_AGENT) == call ||
        __builtin_amdgcn_s_memrealtime() - t0 > DEADLINE_TICKS) {
      if (l == 0) __hip_atomic_store(tmo, call, __ATOMIC_RELAXED, __HIP_MEMORY_SCOPE_AGENT);
      break;
    }
    __builtin_amdgcn_s_sleep(1);
  }
}

template <int GRP>
__device__ __forceinline__ void gemm_role(const Params& P, unsigned call, int gid, char* lds) {
  const int xl = gid & 7, j = gid >> 3;
  const int src = (P.rank + xl) & (NR - 1);
  const bool remote = xl != 0;
  const int mt4 = j >> 3, nt = j & 7;
  const int tid = threadIdx.x, w = __builtin_amdgcn_readfirstlane(tid >> 6), l = tid & 63;
  const int wm = w >> 2, wn = w & 3;

  const unsigned xrs = remote ? KC * 2 : K_ALL * 2;
  const size_t xcs = remote ? CHUNK_B : (size_t)KC * 2;
  const char* xbase = remote ? P.ws + WS_STAGE + (size_t)((call & 1) * NR + src) * SRC_B + (size_t)(mt4 * BM) * xrs
                             : (const char*)P.x + (size_t)(mt4 * BM) * xrs;
  const char* wbase = (const char*)P.w + nt * (BN * 2);
  const unsigned* flags = (const unsigned*)(P.ws + WS_FLAG + (size_t)src * NCH * 128) + mt4 * 4;
  unsigned* tmo = (unsigned*)(P.ws + WS_CTR + 256);

  unsigned xlo[2], wlo[2];
  for (int i = 0; i < 2; ++i) {
    const unsigned row = (2 * w + i) * 8 + (l >> 3);
    xlo[i] = row * xrs + ((((unsigned)l & 7) ^ ((row >> 1) & 7)) << 4);
    const unsigned k = 4 * w + 2 * i + (l >> 5);
    const unsigned cb = (l & 31) * 16;
    const unsigned sw = ((k & 3) << 6) | (((k >> 3) & 1) << 5) | (((k >> 2) & 1) << 4);
    wlo[i] = k * (N_LOC * 2) + (cb ^ sw);
  }
  auto gl_w = [&](int slice, int slot, int i) {
    const char* g = wbase + (size_t)slice * (BK * N_LOC * 2);
    __builtin_amdgcn_global_load_lds((const unsigned*)(g + wlo[i]), (unsigned*)(lds + LDS_W0 + slot * WSL + (2 * w + i) * 1024), 16, 0, 0);
  };
  auto gl_x = [&](int unit, int slot, int i) {
    const char* g = xbase + (size_t)(unit >> 2) * xcs + (unit & 3) * 128;
    __builtin_amdgcn_global_load_lds((const unsigned*)(g + xlo[i]), (unsigned*)(lds + LDS_X0 + slot * XUN + (2 * w + i) * 1024), 16, 0, 17);
  };
  unsigned xro[2];
  {
    const unsigned r = wm * 64 + (l & 15);
    for (int par = 0; par < 2; ++par) xro[par] = LDS_X0 + r * 128 + ((((unsigned)(4 * par + (l >> 4))) ^ ((r >> 1) & 7)) << 4);
  }
  unsigned wro[4][2];
  {
    const unsigned g = l >> 4, q = (l & 15) >> 2, p = l & 3;
    const unsigned L0 = (wn * 128 + (p >> 1) * 16) ^ ((q << 6) | ((g & 1) << 5));
    for (int nb = 0; nb < 4; ++nb) for (int t = 0; t < 2; ++t)
      wro[nb][t] = LDS_W0 + (8 * g + 4 * t + q) * 512 + 8 * (p & 1) + (L0 ^ (nb << 5) ^ (t << 4));
  }
  f32x4 acc[4][4];
  for (int a = 0; a < 4; ++a) for (int b = 0; b < 4; ++b) acc[a][b] = f32x4{0.f, 0.f, 0.f, 0.f};

  struct Frag { bf16x4 wl[4], wh[4]; bf16x8 x[4]; };
  #define RD_W(f, nb, t, so) asm volatile("ds_read_b64_tr_b16 %0, %1" : "=v"((t) ? f.wh[nb] : f.wl[nb]) : "v"(wro[nb][t] + (so)))
  #define RD_X(f, mb, xo) asm volatile("ds_read_b128 %0, %1 offset:%2" : "=v"(f.x[mb]) : "v"(xo), "i"((mb) * 2048))
  auto pin_wait = [&](Frag& f) {
    asm volatile("s_waitcnt lgkmcnt(0)" : "+v"(f.wl[0]), "+v"(f.wl[1]), "+v"(f.wl[2]), "+v"(f.wl[3]),
                 "+v"(f.wh[0]), "+v"(f.wh[1]), "+v"(f.wh[2]), "+v"(f.wh[3]),
                 "+v"(f.x[0]), "+v"(f.x[1]), "+v"(f.x[2]), "+v"(f.x[3]) :: "memory");
  };
  #define G(nb, mb) acc[nb][mb] = __builtin_amdgcn_mfma_f32_16x16x32_bf16(__builtin_shufflevector(cur.wl[nb], cur.wh[nb], 0, 1, 2, 3, 4, 5, 6, 7), cur.x[mb], acc[nb][mb], 0, 0, 0)

  Frag fa, fb;
  if (remote && w == 0) wait_flags(flags, call, tmo, l);
  BAR();
  gl_w(0, 0, 0); gl_w(0, 0, 1); gl_x(0, 0, 0); gl_x(0, 0, 1);
  gl_w(1, 1, 0); gl_w(1, 1, 1);
  gl_w(2, 2, 0); gl_w(2, 2, 1); gl_x(1, 1, 0); gl_x(1, 1, 1);
  WAIT_V(6);
  BAR();
  for (int nb = 0; nb < 4; ++nb) { RD_W(fa, nb, 0, 0); RD_W(fa, nb, 1, 0); }
  for (int mb = 0; mb < 4; ++mb) RD_X(fa, mb, xro[0]);
  int ws = 0;

  auto body = [&](Frag& cur, Frag& nxt, int p, auto PARC, auto GLC, auto RDC, auto WVC, auto PLC) {
    constexpr int PAR = decltype(PARC)::value, L = decltype(GLC)::value, R = decltype(RDC)::value;
    constexpr int WV = decltype(WVC)::value, PL = decltype(PLC)::value;
    pin_wait(cur);
    WAIT_V(WV);
    if (PL) { if (remote && w == 0) wait_flags(flags + (((p + 3) >> 3) * 32), call, tmo, l); }
    SB(); BAR(); SB();
    const int wsn = (ws == 2) ? 0 : ws + 1;
    const unsigned so = wsn * WSL;
    const unsigned xo = xro[PAR ^ 1] + ((((p + 1) >> 1) & 1) * XUN);
    if (GRP == 0) {
      G(0, 0); G(0, 1); SB();
      if (L) gl_w(p + 3, ws, 0);
      SB(); G(0, 2); G(0, 3); SB();
      if (L) gl_w(p + 3, ws, 1);
      SB(); G(1, 0); SB();
      if (L && PAR) { gl_x((p + 3) >> 1, ((p + 3) >> 1) & 1, 0); }
      SB(); G(1, 1); SB();
      if (L && PAR) { gl_x((p + 3) >> 1, ((p + 3) >> 1) & 1, 1); }
      SB(); G(1, 2); SB();
      if (R) RD_W(nxt, 0, 0, so); SB(); G(1, 3); SB();
      if (R) RD_W(nxt, 0, 1, so); SB(); G(2, 0); SB();
      if (R) RD_W(nxt, 1, 0, so); SB(); G(2, 1); SB();
      if (R) RD_W(nxt, 1, 1, so); SB(); G(2, 2); SB();
      if (R) RD_W(nxt, 2, 0, so); SB(); G(2, 3); SB();
      if (R) RD_W(nxt, 2, 1, so); SB(); G(3, 0); SB();
      if (R) { RD_W(nxt, 3, 0, so); RD_X(nxt, 0, xo); } SB(); G(3, 1); SB();
      if (R) { RD_W(nxt, 3, 1, so); RD_X(nxt, 1, xo); } SB(); G(3, 2); SB();
      if (R) { RD_X(nxt, 2, xo); RD_X(nxt, 3, xo); } SB(); G(3, 3); SB();
    } else {
      if (R) RD_W(nxt, 0, 0, so); SB(); G(0, 0); SB();
      if (R) RD_W(nxt, 0, 1, so); SB(); G(0, 1); SB();
      if (R) RD_W(nxt, 1, 0, so); SB(); G(0, 2); SB();
      if (R) RD_W(nxt, 1, 1, so); SB(); G(0, 3); SB();
      if (R) RD_W(nxt, 2, 0, so); SB(); G(1, 0); SB();
      if (R) RD_W(nxt, 2, 1, so); SB(); G(1, 1); SB();
      if (R) { RD_W(nxt, 3, 0, so); RD_X(nxt, 0, xo); } SB(); G(1, 2); SB();
      if (R) { RD_W(nxt, 3, 1, so); RD_X(nxt, 1, xo); } SB(); G(1, 3); SB();
      if (R) { RD_X(nxt, 2, xo); RD_X(nxt, 3, xo); } SB(); G(2, 0); G(2, 1); SB();
      if (L) gl_w(p + 3, ws, 0);
      SB(); G(2, 2); G(2, 3); SB();
      if (L) gl_w(p + 3, ws, 1);
      SB(); G(3, 0); SB();
      if (L && PAR) { gl_x((p + 3) >> 1, ((p + 3) >> 1) & 1, 0); }
      SB(); G(3, 1); SB();
      if (L && PAR) { gl_x((p + 3) >> 1, ((p + 3) >> 1) & 1, 1); }
      SB(); G(3, 2); G(3, 3); SB();
    }
    ws = wsn;
  };
  using I0 = std::integral_constant<int, 0>; using I1 = std::integral_constant<int, 1>;
  using I2 = std::integral_constant<int, 2>; using I4 = std::integral_constant<int, 4>;
  for (int c = 0; c < NCH - 1; ++c) {
    const int p = c * SPC;
    body(fa, fb, p + 0, I0{}, I1{}, I1{}, I4{}, I0{});
    body(fb, fa, p + 1, I1{}, I1{}, I1{}, I2{}, I0{});
    body(fa, fb, p + 2, I0{}, I1{}, I1{}, I4{}, I0{});
    body(fb, fa, p + 3, I1{}, I1{}, I1{}, I2{}, I0{});
    body(fa, fb, p + 4, I0{}, I1{}, I1{}, I4{}, I0{});
    body(fb, fa, p + 5, I1{}, I1{}, I1{}, I2{}, I1{});
    body(fa, fb, p + 6, I0{}, I1{}, I1{}, I4{}, I0{});
    body(fb, fa, p + 7, I1{}, I1{}, I1{}, I2{}, I0{});
  }
  {
    const int p = (NCH - 1) * SPC;
    body(fa, fb, p + 0, I0{}, I1{}, I1{}, I4{}, I0{});
    body(fb, fa, p + 1, I1{}, I1{}, I1{}, I2{}, I0{});
    body(fa, fb, p + 2, I0{}, I1{}, I1{}, I4{}, I0{});
    body(fb, fa, p + 3, I1{}, I1{}, I1{}, I2{}, I0{});
    body(fa, fb, p + 4, I0{}, I1{}, I1{}, I4{}, I0{});
    body(fb, fa, p + 5, I1{}, I0{}, I1{}, I2{}, I0{});
    body(fa, fb, p + 6, I0{}, I0{}, I1{}, I0{}, I0{});
    body(fb, fa, p + 7, I1{}, I0{}, I0{}, I0{}, I0{});
  }
  {
    const bool bad = remote && __hip_atomic_load(tmo, __ATOMIC_RELAXED, __HIP_MEMORY_SCOPE_AGENT) == call;
    char* ep = lds + w * 8192;
    const unsigned mrow = l & 15, g = l >> 4;
    for (int nb = 0; nb < 4; ++nb) for (int mb = 0; mb < 4; ++mb) {
      const f32x4 a = acc[nb][mb];
      u32x2 v; v.x = pack_bf16(silu_f(a[0]), silu_f(a[1])); v.y = pack_bf16(silu_f(a[2]), silu_f(a[3]));
      if (bad) { v.x = 0x7FC07FC0u; v.y = 0x7FC07FC0u; }
      const unsigned row = mb * 16 + mrow, gran = nb * 4 + g;
      *reinterpret_cast<u32x2*>(ep + row * 128 + ((gran ^ (row & 15)) << 3)) = v;
    }
    WAIT_L(0);
    unsigned short* obase = P.out + (size_t)(src * M_LOC + mt4 * BM + wm * 64) * N_LOC + nt * BN + wn * 64;
    for (int i = 0; i < 8; ++i) {
      const unsigned row = i * 8 + (l >> 3), c = l & 7, x = row & 15;
      u32x4 v = *reinterpret_cast<const u32x4*>(ep + row * 128 + ((c ^ (x >> 1)) << 4));
      if (x & 1) { u32x4 t; t.x = v.z; t.y = v.w; t.z = v.x; t.w = v.y; v = t; }
      *reinterpret_cast<u32x4*>(obase + (size_t)row * N_LOC + c * 8) = v;
    }
  }
  #undef G
  #undef RD_W
  #undef RD_X
}

__global__ void __launch_bounds__(512, 3) fused_ag_gemm_silu(Params P) {
  extern __shared__ __attribute__((aligned(16))) char lds[];
  const int b = blockIdx.x;
  const int tid = threadIdx.x, wv = __builtin_amdgcn_readfirstlane(tid >> 6), l = tid & 63;
  const bool is_push = b < NPUSH;
  if (is_push && wv >= 4) return;
  volatile unsigned* lseq = (volatile unsigned*)(lds + LDS_BYTES - 16);
  if (tid == 0) {
    unsigned* ctr = (unsigned*)(P.ws + WS_CTR + (is_push ? 0 : 128));
    const unsigned t = __hip_atomic_fetch_add(ctr, 1u, __ATOMIC_RELAXED, __HIP_MEMORY_SCOPE_AGENT);
    *lseq = t / (unsigned)(is_push ? NPUSH : NGEMM) + 1u;
  }
  __syncthreads();
  const unsigned call = __builtin_amdgcn_readfirstlane(*lseq);
  __syncthreads();
  if (is_push) {
    pusher(P, call, b, wv, l);
  } else {
    const int gid = b - NPUSH;
    if (wv < 4) gemm_role<0>(P, call, gid, lds);
    else gemm_role<1>(P, call, gid, lds);
  }
}

}

extern "C" void kernel_launch(int rank, int nranks, ncclComm_t comm,
                              hipStream_t stream,
                              void* const* d_in, const int* in_sizes, int n_in,
                              void* d_out, int out_size,
                              const int* in_dtypes, int out_dtype,
                              void* const* d_out_all,
                              void* const* d_ws_all, size_t ws_bytes) {
  (void)comm; (void)in_dtypes; (void)out_dtype; (void)d_out_all;
  if (nranks != NR || n_in < 2 || in_sizes[0] != M_LOC * K_ALL || in_sizes[1] != K_ALL * N_LOC ||
      out_size != M_ALL * N_LOC || ws_bytes < WS_NEED) {
    fprintf(stderr, "kernel_launch: unexpected problem shape\n");
    return;
  }
  static bool attr_set = false;
  if (!attr_set) {
    HIP_CHECK(hipFuncSetAttribute((const void*)fused_ag_gemm_silu, hipFuncAttributeMaxDynamicSharedMemorySize, LDS_BYTES));
    attr_set = true;
  }
  Params P;
  P.x = (const unsigned short*)d_in[0];
  P.w = (const unsigned short*)d_in[1];
  P.out = (unsigned short*)d_out;
  P.ws = (char*)d_ws_all[rank];
  P.ws_all = d_ws_all;
  P.rank = rank;
  fused_ag_gemm_silu<<<GRID, 512, LDS_BYTES, stream>>>(P);
  HIP_CHECK(hipGetLastError());
}
```

```cpp
#include <hip/hip_runtime.h>
#include <hip/hip_bf16.h>
#include <rccl/rccl.h>
#include <cstdio>
#include <type_traits>

#define HIP_CHECK(expr) do { hipError_t _e = (expr); if (_e != hipSuccess) { \
    fprintf(stderr, "HIP error %s at %s:%d\n", hipGetErrorString(_e), __FILE__, __LINE__); \
    return; } } while (0)

namespace {

typedef short bf16x8 __attribute__((ext_vector_type(8)));
typedef short bf16x4 __attribute__((ext_vector_type(4)));
typedef float f32x4 __attribute__((ext_vector_type(4)));
typedef unsigned u32x2 __attribute__((ext_vector_type(2)));
typedef unsigned u32x4 __attribute__((ext_vector_type(4)));

constexpr int NR = 8;
constexpr int M_ALL = 4096, K_ALL = 4096, N_LOC = 2048, M_LOC = 512;
constexpr int BM = 160, BN = 256, BK = 32;
constexpr int MB = BM / 32;
constexpr int NSLICE = K_ALL / BK;
constexpr int WSL = BK * BN * 2;
constexpr int XUN = BM * 64 * 2;
constexpr int LDS_W0 = 0, LDS_X0 = 3 * WSL;
constexpr int LDS_BYTES = 3 * WSL + 2 * XUN;
#define KCH 128
constexpr int KC = KCH, NCH = K_ALL / KC;
constexpr int SPC = KC / BK, UPC = KC / 64;
constexpr int RB = KC * 2;
constexpr size_t CHUNK_B = (size_t)M_LOC * KC * 2;
constexpr size_t SRC_B = CHUNK_B * NCH;

#define PTHR 2
constexpr int BAND_ROWS = 16;
constexpr int NBAND = M_LOC / BAND_ROWS;
constexpr int NPUSH = NBAND;
constexpr int NMT = (M_ALL + BM - 1) / BM;
constexpr int NGEMM = NMT * (N_LOC / BN);
constexpr int GRID = NPUSH + NGEMM;
constexpr int NTHREADS = 768;
static_assert(NPUSH % 8 == 0 && NGEMM % 8 == 0, "ticket shards assume multiples of 8");

constexpr size_t WS_STAGE = 0;
constexpr size_t WS_FLAG = 2 * NR * SRC_B;
constexpr size_t WS_CTR = WS_FLAG + (size_t)NCH * 1024;
constexpr size_t WS_NEED = WS_CTR + 4096;
#define STAMP(i) do { } while (0)
constexpr unsigned long long DEADLINE_TICKS = 50000000ull;

struct Params {
  const unsigned short* x;
  const unsigned short* w;
  unsigned short* out;
  char* ws;
  void* const* ws_all;
  int rank;
};

#define WAIT_V(n) asm volatile("s_waitcnt vmcnt(%0)" :: "i"(n) : "memory")
#define WAIT_L(n) asm volatile("s_waitcnt lgkmcnt(%0)" :: "i"(n) : "memory")
#define BAR() asm volatile("s_barrier" ::: "memory")
#define SB() __builtin_amdgcn_sched_barrier(0)

__device__ __forceinline__ int lane_now() {
  int z = 0; asm volatile("" : "+v"(z));
  return __builtin_amdgcn_mbcnt_hi(~0u, __builtin_amdgcn_mbcnt_lo(~0u, (unsigned)z));
}
__device__ __forceinline__ float silu_f(float y) {
  const float c = fminf(fmaxf(y, -60.f), 60.f);
  const float e = __expf(-c);
  return y * __builtin_amdgcn_rcpf(1.f + e);
}
__device__ __forceinline__ unsigned pack_bf16(float a, float b) {
  __hip_bfloat162 v = __float22bfloat162_rn(make_float2(a, b));
  return *reinterpret_cast<unsigned*>(&v);
}

#define P_LD(r, vo, sb) asm volatile("global_load_dwordx4 %0, %1, %2" : "=v"(r) : "v"(vo), "s"(sb) : "memory")
#define P_ST(vo, r, sb, imm) asm volatile("global_store_dwordx4 %0, %1, %2 offset:%3 sc0 sc1\n\ts_nop 1" :: "v"(vo), "v"(r), "s"(sb), "i"(imm) : "memory")
#define P_PIN(n, R) asm volatile("s_waitcnt vmcnt(%4)" : "+v"(R[0]), "+v"(R[1]), "+v"(R[2]), "+v"(R[3]) : "i"(n) : "memory")

constexpr int PNI = BAND_ROWS * RB / 1024;
constexpr int PRPI = 1024 / RB;
static_assert(PNI >= 1 && PNI <= 8, "band-chunk piece must be 1..8 KiB");

__device__ __forceinline__ void pusher(const Params& P, unsigned call, int band, int v, int l) {
  const int dst = (P.rank + v) & (NR - 1);
  char* pws;
  {
    const unsigned long long a = (unsigned long long)P.ws_all[dst];
    const unsigned lo = __builtin_amdgcn_readfirstlane((unsigned)a), hi = __builtin_amdgcn_readfirstlane((unsigned)(a >> 32));
    pws = (char*)(((unsigned long long)hi << 32) | lo);
  }
  const char* sb = (const char*)P.x + (size_t)(band * BAND_ROWS) * (K_ALL * 2);
  char* db = pws + WS_STAGE + (size_t)((call & 1) * NR + P.rank) * SRC_B + (size_t)(band * BAND_ROWS) * RB;
  unsigned* fl = (unsigned*)(pws + WS_FLAG) + P.rank * NBAND + band;
  constexpr int LPR = RB / 16;
  const unsigned so0 = (unsigned)(l / LPR) * (K_ALL * 2) + (l % LPR) * 16;
  const unsigned do0 = l * 16;
  u32x4 A[PNI], B[PNI];
  auto ld = [&](u32x4* R, int c) {
    for (int i = 0; i < PNI; ++i) P_LD(R[i], so0, sb + (size_t)c * RB + (size_t)(i * PRPI) * (K_ALL * 2));
  };
  auto st = [&](u32x4* R, int c) {
    char* d = db + (size_t)c * CHUNK_B;
    for (int i = 0; i < PNI; ++i) {
      P_ST(do0, R[i], d + (i >> 2) * 4096, (i & 3) * 1024);
      WAIT_V(PTHR);
    }
  };
  auto flag = [&](int c) {
    if (l == 0) __hip_atomic_store(fl + c * 256, call, __ATOMIC_RELAXED, __HIP_MEMORY_SCOPE_SYSTEM);
  };
  auto pin = [&](u32x4* R) { for (int i = 0; i < PNI; ++i) asm volatile("" : "+v"(R[i])); };
  ld(A, 0); ld(B, 1);
  for (int c = 0; c < NCH; c += 2) {
    WAIT_V(PNI); pin(A);
    st(A, c);
    if (c + 2 < NCH) { ld(A, c + 2); WAIT_V(PNI); } else { WAIT_V(0); }
    pin(B);
    flag(c);
    st(B, c + 1);
    if (c + 3 < NCH) { ld(B, c + 3); WAIT_V(PNI); } else { WAIT_V(0); }
    flag(c + 1);
    if (v == 1 && (c & 1) == 0 && NCH <= 32) STAMP(1 + (c >> 1));
  }
}

__device__ __forceinline__ void wait_flags(const unsigned* fl, int nb, unsigned call, unsigned* tmo) {
  const int l = lane_now();
  const unsigned long long t0 = __builtin_amdgcn_s_memrealtime();
  for (;;) {
    unsigned v = call;
    if (l < nb) v = __hip_atomic_load(fl + l, __ATOMIC_RELAXED, __HIP_MEMORY_SCOPE_SYSTEM);
    if (__builtin_amdgcn_ballot_w64(v < call) == 0) break;
    if (__hip_atomic_load(tmo, __ATOMIC_RELAXED, __HIP_MEMORY_SCOPE_AGENT) == call ||
        __builtin_amdgcn_s_memrealtime() - t0 > DEADLINE_TICKS) {
      if (l == 0) __hip_atomic_store(tmo, call, __ATOMIC_RELAXED, __HIP_MEMORY_SCOPE_AGENT);
      break;
    }
    __builtin_amdgcn_s_sleep(1);
  }
}

__device__ __forceinline__ void loader_role(const Params& P, unsigned call, int gid) {
  const int nt = gid & 7, mt = gid >> 3;
  const int tid = threadIdx.x, lw = __builtin_amdgcn_readfirstlane(tid >> 6) - 8, l = tid & 63;
  const char* xbase = P.ws + WS_STAGE + (size_t)(call & 1) * NR * SRC_B;
  const char* wbase = (const char*)P.w + nt * (BN * 2);
  const unsigned* flags = (const unsigned*)(P.ws + WS_FLAG) + mt * (BM / BAND_ROWS);
  const int nflag = min(BM / BAND_ROWS, M_ALL / BAND_ROWS - mt * (BM / BAND_ROWS));
  unsigned* tmo = (unsigned*)(P.ws + WS_CTR + 16 * 128);
  unsigned xlo[5], wlo[4];
  for (int i = 0; i < 5; ++i) {
    const unsigned row = (unsigned)(lw * 5 + i) * 8 + (l >> 3);
    const unsigned R = min((unsigned)(mt * BM) + row, (unsigned)(M_ALL - 1));
    xlo[i] = (R >> 9) * (unsigned)SRC_B + (R & 511) * RB + ((((unsigned)l & 7) ^ ((row >> 1) & 7)) << 4);
  }
  for (int i = 0; i < 4; ++i) {
    const unsigned k = (unsigned)(lw * 4 + i) * 2 + (l >> 5);
    const unsigned cb = (l & 31) * 16;
    const unsigned sw = ((k & 3) << 6) | (((k >> 3) & 1) << 5) | (((k >> 2) & 1) << 4);
    wlo[i] = k * (N_LOC * 2) + (cb ^ sw);
  }
  #define GLDS(vo, sbase, ldsa, POL) asm volatile("s_mov_b32 m0, %2\n\ts_nop 0\n\tglobal_load_lds_dwordx4 %0, %1" POL :: "v"(vo), "s"(sbase), "s"(ldsa) : "memory")
  auto gl_w = [&](int slice, int slot) {
    const char* g = wbase + (size_t)slice * (BK * N_LOC * 2);
    for (int i = 0; i < 4; ++i) GLDS(wlo[i], g, (unsigned)(LDS_W0 + slot * WSL + (lw * 4 + i) * 1024), "");
  };
  auto gl_x = [&](int unit, int slot) {
    const char* g = xbase + (size_t)(unit / UPC) * CHUNK_B + (unit % UPC) * 128;
    for (int i = 0; i < 5; ++i) GLDS(xlo[i], g, (unsigned)(LDS_X0 + slot * XUN + (lw * 5 + i) * 1024), " sc0 sc1");
  };
  if (lw == 0) { wait_flags(flags, nflag, call, tmo); STAMP(1); }
  BAR();
  gl_w(0, 0); gl_x(0, 0); gl_w(1, 1); gl_w(2, 2); gl_x(1, 1);
  WAIT_V(13);
  BAR();
  int ws = 0;
  for (int p = 0; p < NSLICE; ++p) {
    if (p + 2 >= NSLICE) { WAIT_V(0); } else if (p & 1) { WAIT_V(4); } else { WAIT_V(9); }
    if (((p + 3) % SPC) == 0 && p + 3 < NSLICE && lw == 0) { wait_flags(flags + ((p + 3) / SPC) * 256, nflag, call, tmo); if (((p + 3) & 7) == 0) STAMP(1 + ((p + 3) >> 3)); }
    BAR();
    if (p + 3 < NSLICE) {
      gl_w(p + 3, ws);
      if (p & 1) gl_x((p + 3) >> 1, ((p + 3) >> 1) & 1);
    }
    ws = (ws == 2) ? 0 : ws + 1;
  }
  #undef GLDS
}

template <int GRP>
__device__ __forceinline__ void gemm_role(const Params& P, unsigned call, int gid, char* lds) {
  const int nt = gid & 7, mt = gid >> 3;
  const int tid = threadIdx.x, w = __builtin_amdgcn_readfirstlane(tid >> 6), l = tid & 63;
  constexpr int wm = GRP;
  const int wn = w & 3;
  unsigned* tmo = (unsigned*)(P.ws + WS_CTR + 16 * 128);

  unsigned xr0;
  {
    const unsigned r = wm * (BM / 2) + (l & 15);
    xr0 = LDS_X0 + r * 128 + ((((unsigned)(l >> 4)) ^ ((r >> 1) & 7)) << 4);
  }
  unsigned wr0;
  {
    const unsigned g = l >> 4, q = (l & 15) >> 2, p = l & 3;
    wr0 = LDS_W0 + (8 * g + q) * 512 + 8 * (p & 1) + ((wn * 128 + (p >> 1) * 16) ^ ((q << 6) | ((g & 1) << 5)));
  }
  f32x4 acc[4][MB];
  for (int a = 0; a < 4; ++a) for (int b = 0; b < MB; ++b) acc[a][b] = f32x4{0.f, 0.f, 0.f, 0.f};

  struct Frag { bf16x4 wl[4], wh[4]; bf16x8 x[MB]; };
  #define RD_W(f, nb, t, wb) do { unsigned _t; \
      asm volatile("v_xor_b32 %1, %3, %2\n\tds_read_b64_tr_b16 %0, %1 offset:%4" : "=v"((t) ? f.wh[nb] : f.wl[nb]), "=&v"(_t) \
                   : "v"(wb), "i"(((nb) << 5) | ((t) << 4)), "i"((t) * 2048)); } while (0)
  #define RD_X(f, mb, xo) asm volatile("ds_read_b128 %0, %1 offset:%2" : "=v"(f.x[mb]) : "v"(xo), "i"((mb) * 2048))
  auto pin_wait = [&](Frag& f) {
    asm volatile("s_waitcnt lgkmcnt(0)" : "+v"(f.wl[0]), "+v"(f.wl[1]), "+v"(f.wl[2]), "+v"(f.wl[3]),
                 "+v"(f.wh[0]), "+v"(f.wh[1]), "+v"(f.wh[2]), "+v"(f.wh[3]),
                 "+v"(f.x[0]), "+v"(f.x[1]), "+v"(f.x[2]), "+v"(f.x[3]), "+v"(f.x[4]) :: "memory");
  };
  #define G(nb, mb) acc[nb][mb] = __builtin_amdgcn_mfma_f32_16x16x32_bf16(__builtin_shufflevector(cur.wl[nb], cur.wh[nb], 0, 1, 2, 3, 4, 5, 6, 7), cur.x[mb], acc[nb][mb], 0, 0, 0)

  Frag fa, fb;
  BAR();
  BAR();
  for (int nb = 0; nb < 4; ++nb) { RD_W(fa, nb, 0, wr0); RD_W(fa, nb, 1, wr0); }
  for (int mb = 0; mb < MB; ++mb) RD_X(fa, mb, xr0);
  int ws = 0;

  auto body = [&](Frag& cur, Frag& nxt, int p, auto PARC, auto RDC) {
    constexpr int PAR = decltype(PARC)::value, R = decltype(RDC)::value;
    pin_wait(cur);
    SB(); BAR(); SB();
    const int wsn = (ws == 2) ? 0 : ws + 1;
    const unsigned so = wr0 + wsn * WSL;
    const unsigned xo = (xr0 ^ (unsigned)((PAR ^ 1) * 64)) + ((((p + 1) >> 1) & 1) * XUN);
    if (R) RD_W(nxt, 0, 0, so); SB(); G(0, 0); SB();
    if (R) RD_W(nxt, 0, 1, so); SB(); G(0, 1); SB();
    if (R) RD_W(nxt, 1, 0, so); SB(); G(0, 2); SB();
    if (R) RD_W(nxt, 1, 1, so); SB(); G(0, 3); SB();
    if (R) RD_W(nxt, 2, 0, so); SB(); G(0, 4); SB();
    if (R) RD_W(nxt, 2, 1, so); SB(); G(1, 0); SB();
    if (R) RD_W(nxt, 3, 0, so); SB(); G(1, 1); SB();
    if (R) RD_W(nxt, 3, 1, so); SB(); G(1, 2); SB();
    if (R) RD_X(nxt, 0, xo); SB(); G(1, 3); SB();
    if (R) RD_X(nxt, 1, xo); SB(); G(1, 4); SB();
    if (R) RD_X(nxt, 2, xo); SB(); G(2, 0); SB();
    if (R) RD_X(nxt, 3, xo); SB(); G(2, 1); SB();
    if (R) RD_X(nxt, 4, xo); SB(); G(2, 2); G(2, 3); G(2, 4); G(3, 0); G(3, 1); G(3, 2); G(3, 3); G(3, 4); SB();
    ws = wsn;
  };
  using I0 = std::integral_constant<int, 0>; using I1 = std::integral_constant<int, 1>;
  for (int p = 0; p < NSLICE - 2; p += 2) {
    body(fa, fb, p, I0{}, I1{});
    body(fb, fa, p + 1, I1{}, I1{});
  }
  body(fa, fb, NSLICE - 2, I0{}, I1{});
  body(fb, fa, NSLICE - 1, I1{}, I0{});
  if (w == 0) STAMP(17);
  {
    const bool bad = __hip_atomic_load(tmo, __ATOMIC_RELAXED, __HIP_MEMORY_SCOPE_AGENT) == call;
    char* ep = lds + w * (MB * 16 * 128);
    const int l = lane_now();
    const unsigned mrow = l & 15, g = l >> 4;
    for (int nb = 0; nb < 4; ++nb) for (int mb = 0; mb < MB; ++mb) {
      const f32x4 a = acc[nb][mb];
      u32x2 v; v.x = pack_bf16(silu_f(a[0]), silu_f(a[1])); v.y = pack_bf16(silu_f(a[2]), silu_f(a[3]));
      if (bad) { v.x = 0x7FC07FC0u; v.y = 0x7FC07FC0u; }
      const unsigned row = mb * 16 + mrow, gran = nb * 4 + g;
      *reinterpret_cast<u32x2*>(ep + row * 128 + ((gran ^ (row & 15)) << 3)) = v;
    }
    WAIT_L(0);
    const int row0 = mt * BM + wm * (BM / 2);
    unsigned short* obase = P.out + (size_t)row0 * N_LOC + nt * BN + wn * 64;
    for (int i = 0; i < 2 * MB; ++i) {
      const unsigned row = i * 8 + (l >> 3), c = l & 7, x = row & 15;
      u32x4 v = *reinterpret_cast<const u32x4*>(ep + row * 128 + ((c ^ (x >> 1)) << 4));
      if (x & 1) { u32x4 t; t.x = v.z; t.y = v.w; t.z = v.x; t.w = v.y; v = t; }
      if (row0 + (int)row < M_ALL) *reinterpret_cast<u32x4*>(obase + (size_t)row * N_LOC + c * 8) = v;
    }
  }
  if (w == 0) { WAIT_V(0); STAMP(18); }
  #undef G
  #undef RD_W
  #undef RD_X
}

__global__ void __launch_bounds__(NTHREADS, 3) fused_ag_gemm_silu(Params P) {
  extern __shared__ __attribute__((aligned(16))) char lds[];
  const int b = blockIdx.x;
  const int tid = threadIdx.x, wv = __builtin_amdgcn_readfirstlane(tid >> 6), l = tid & 63;
  const bool is_push = b < NPUSH;
  volatile unsigned* lseq = (volatile unsigned*)(lds + LDS_BYTES - 16);
  if (tid == 0) {
    unsigned* ctr = (unsigned*)(P.ws + WS_CTR + ((is_push ? 0 : 8) + (b & 7)) * 128);
    const unsigned t = __hip_atomic_fetch_add(ctr, 1u, __ATOMIC_RELAXED, __HIP_MEMORY_SCOPE_AGENT);
    *lseq = t / (unsigned)(is_push ? NPUSH / 8 : NGEMM / 8) + 1u;
  }
  __syncthreads();
  const unsigned call = __builtin_amdgcn_readfirstlane(*lseq);
  __syncthreads();
  if (wv == 0) STAMP(0);
  if (is_push) {
    if (wv < NR) pusher(P, call, b, wv, l);
  } else {
    const int gid = b - NPUSH;
    if (wv < 4) gemm_role<0>(P, call, gid, lds);
    else if (wv < 8) gemm_role<1>(P, call, gid, lds);
    else loader_role(P, call, gid);
  }
}

}

extern "C" void kernel_launch(int rank, int nranks, ncclComm_t comm,
                              hipStream_t stream,
                              void* const* d_in, const int* in_sizes, int n_in,
                              void* d_out, int out_size,
                              const int* in_dtypes, int out_dtype,
                              void* const* d_out_all,
                              void* const* d_ws_all, size_t ws_bytes) {
  (void)comm; (void)in_dtypes; (void)out_dtype; (void)d_out_all;
  if (nranks != NR || n_in < 2 || in_sizes[0] != M_LOC * K_ALL || in_sizes[1] != K_ALL * N_LOC ||
      out_size != M_ALL * N_LOC || ws_bytes < WS_NEED) {
    fprintf(stderr, "kernel_launch: unexpected problem shape\n");
    return;
  }
  static bool attr_set = false;
  if (!attr_set) {
    HIP_CHECK(hipFuncSetAttribute((const void*)fused_ag_gemm_silu, hipFuncAttributeMaxDynamicSharedMemorySize, LDS_BYTES));
    attr_set = true;
  }
  Params P;
  P.x = (const unsigned short*)d_in[0];
  P.w = (const unsigned short*)d_in[1];
  P.out = (unsigned short*)d_out;
  P.ws = (char*)d_ws_all[rank];
  P.ws_all = d_ws_all;
  P.rank = rank;
  fused_ag_gemm_silu<<<GRID, NTHREADS, LDS_BYTES, stream>>>(P);
  HIP_CHECK(hipGetLastError());
}
```

```cpp
#include <hip/hip_runtime.h>
#include <hip/hip_bf16.h>
#include <rccl/rccl.h>
#include <cstdio>
#include <type_traits>

#define HIP_CHECK(expr) do { hipError_t _e = (expr); if (_e != hipSuccess) { \
    fprintf(stderr, "HIP error %s at %s:%d\n", hipGetErrorString(_e), __FILE__, __LINE__); \
    return; } } while (0)

namespace {

typedef short bf16x8 __attribute__((ext_vector_type(8)));
typedef short bf16x4 __attribute__((ext_vector_type(4)));
typedef float f32x4 __attribute__((ext_vector_type(4)));
typedef unsigned u32x2 __attribute__((ext_vector_type(2)));
typedef unsigned u32x4 __attribute__((ext_vector_type(4)));

constexpr int NR = 8;
constexpr int M_ALL = 4096, K_ALL = 4096, N_LOC = 2048, M_LOC = 512;
constexpr int BM = 160, BN = 256, BK = 32;
constexpr int MB = BM / 32;
constexpr int NSLICE = K_ALL / BK;
constexpr int WSL = BK * BN * 2;
constexpr int XUN = BM * 64 * 2;
constexpr int LDS_W0 = 0, LDS_X0 = 3 * WSL;
constexpr int LDS_BYTES = 3 * WSL + 2 * XUN;
#define KCH 128
constexpr int KC = KCH, NCH = K_ALL / KC;
constexpr int SPC = KC / BK, UPC = KC / 64;
constexpr int RB = KC * 2;
constexpr size_t CHUNK_B = (size_t)M_LOC * KC * 2;
constexpr size_t SRC_B = CHUNK_B * NCH;

#define PTHR 2
constexpr int BAND_ROWS = 16;
constexpr int NBAND = M_LOC / BAND_ROWS;
constexpr int NPUSH = NBAND;
constexpr int NMT = (M_ALL + BM - 1) / BM;
constexpr int NGEMM = NMT * (N_LOC / BN);
constexpr int GRID = NPUSH + NGEMM;
constexpr int NTHREADS = 768;
static_assert(NPUSH % 8 == 0 && NGEMM % 8 == 0, "ticket shards assume multiples of 8");

constexpr size_t WS_STAGE = 0;
constexpr size_t WS_FLAG = 2 * NR * SRC_B;
constexpr size_t WS_CTR = WS_FLAG + (size_t)NCH * 1024;
constexpr size_t WS_NEED = WS_CTR + 4096;
#define STAMP(i) do { } while (0)
constexpr unsigned long long DEADLINE_TICKS = 50000000ull;

struct Params {
  const unsigned short* x;
  const unsigned short* w;
  unsigned short* out;
  char* ws;
  void* const* ws_all;
  int rank;
};

#define WAIT_V(n) asm volatile("s_waitcnt vmcnt(%0)" :: "i"(n) : "memory")
#define WAIT_L(n) asm volatile("s_waitcnt lgkmcnt(%0)" :: "i"(n) : "memory")
#define BAR() asm volatile("s_barrier" ::: "memory")
#define SB() __builtin_amdgcn_sched_barrier(0)

__device__ __forceinline__ int lane_now() {
  int z = 0; asm volatile("" : "+v"(z));
  return __builtin_amdgcn_mbcnt_hi(~0u, __builtin_amdgcn_mbcnt_lo(~0u, (unsigned)z));
}
__device__ __forceinline__ float silu_f(float y) {
  const float c = fminf(fmaxf(y, -60.f), 60.f);
  const float e = __expf(-c);
  return y * __builtin_amdgcn_rcpf(1.f + e);
}
__device__ __forceinline__ unsigned pack_bf16(float a, float b) {
  __hip_bfloat162 v = __float22bfloat162_rn(make_float2(a, b));
  return *reinterpret_cast<unsigned*>(&v);
}

#define P_ST(vo, r, sb, imm) asm volatile("global_store_dwordx4 %0, %1, %2 offset:%3 sc0 sc1\n\ts_nop 1" :: "v"(vo), "v"(r), "s"(sb), "i"(imm) : "memory")

constexpr int PNI = BAND_ROWS * RB / 1024;
constexpr int PRPI = 1024 / RB;
static_assert(PNI >= 1 && PNI <= 8, "band-chunk piece must be 1..8 KiB");

__device__ __forceinline__ void pusher(const Params& P, unsigned call, int band, int v, int l) {
  const int dst = (P.rank + v) & (NR - 1);
  char* pws;
  {
    const unsigned long long a = (unsigned long long)P.ws_all[dst];
    const unsigned lo = __builtin_amdgcn_readfirstlane((unsigned)a), hi = __builtin_amdgcn_readfirstlane((unsigned)(a >> 32));
    pws = (char*)(((unsigned long long)hi << 32) | lo);
  }
  const char* sb = (const char*)P.x + (size_t)(band * BAND_ROWS) * (K_ALL * 2);
  char* db = pws + WS_STAGE + (size_t)((call & 1) * NR + P.rank) * SRC_B + (size_t)(band * BAND_ROWS) * RB;
  unsigned* fl = (unsigned*)(pws + WS_FLAG) + P.rank * NBAND + band;
  constexpr int LPR = RB / 16;
  const unsigned so0 = (unsigned)(l / LPR) * (K_ALL * 2) + (l % LPR) * 16;
  const unsigned do0 = l * 16;
  u32x4 A[PNI], B[PNI];
  #define P_LD(r, vo, sb) asm volatile("global_load_dwordx4 %0, %1, %2" : "=v"(r) : "v"(vo), "s"(sb) : "memory")
  auto ld = [&](u32x4* R, int c) {
    for (int i = 0; i < PNI; ++i) P_LD(R[i], so0, sb + (size_t)c * RB + (size_t)(i * PRPI) * (K_ALL * 2));
  };
  auto pin = [&](u32x4* R) { for (int i = 0; i < PNI; ++i) asm volatile("" : "+v"(R[i])); };
  auto st = [&](u32x4* R, int c) {
    char* d = db + (size_t)c * CHUNK_B;
    for (int i = 0; i < PNI; ++i) {
      P_ST(do0, R[i], d + (i >> 2) * 4096, (i & 3) * 1024);
      WAIT_V(PTHR);
    }
  };
  auto flag = [&](int c) {
    if (l == 0) __hip_atomic_store(fl + c * 256, call, __ATOMIC_RELAXED, __HIP_MEMORY_SCOPE_SYSTEM);
  };
  ld(A, 0); ld(B, 1);
  for (int c = 0; c < NCH - 2; c += 2) {
    WAIT_V(PNI); pin(A);
    st(A, c); ld(A, c + 2);
    WAIT_V(PNI); pin(B);
    flag(c);
    st(B, c + 1); ld(B, c + 3);
    WAIT_V(PNI);
    flag(c + 1);
    if (v == 1 && NCH <= 32) STAMP(1 + (c >> 1));
  }
  WAIT_V(PNI); pin(A);
  st(A, NCH - 2);
  WAIT_V(0); pin(B);
  flag(NCH - 2);
  st(B, NCH - 1);
  WAIT_V(0);
  flag(NCH - 1);
  #undef P_LD
}

__device__ __forceinline__ void wait_flags(const unsigned* fl, int nb, unsigned call, unsigned* tmo) {
  const int l = lane_now();
  const unsigned long long t0 = __builtin_amdgcn_s_memrealtime();
  for (;;) {
    unsigned v = call;
    if (l < nb) v = __hip_atomic_load(fl + l, __ATOMIC_RELAXED, __HIP_MEMORY_SCOPE_SYSTEM);
    if (__builtin_amdgcn_ballot_w64(v < call) == 0) break;
    if (__hip_atomic_load(tmo, __ATOMIC_RELAXED, __HIP_MEMORY_SCOPE_AGENT) == call ||
        __builtin_amdgcn_s_memrealtime() - t0 > DEADLINE_TICKS) {
      if (l == 0) __hip_atomic_store(tmo, call, __ATOMIC_RELAXED, __HIP_MEMORY_SCOPE_AGENT);
      break;
    }
    __builtin_amdgcn_s_sleep(1);
  }
}

__device__ __forceinline__ void loader_role(const Params& P, unsigned call, int gid, char* lds) {
  const int nt = gid & 7, mt = gid >> 3;
  const int tid = threadIdx.x, lw = __builtin_amdgcn_readfirstlane(tid >> 6) - 8, l = tid & 63;
  const char* xbase = P.ws + WS_STAGE + (size_t)(call & 1) * NR * SRC_B;
  const char* wbase = (const char*)P.w + nt * (BN * 2);
  const unsigned* flags = (const unsigned*)(P.ws + WS_FLAG) + mt * (BM / BAND_ROWS);
  const int nflag = min(BM / BAND_ROWS, M_ALL / BAND_ROWS - mt * (BM / BAND_ROWS));
  unsigned* tmo = (unsigned*)(P.ws + WS_CTR + 16 * 128);
  unsigned xlo[5], wlo[4];
  for (int i = 0; i < 5; ++i) {
    const unsigned row = (unsigned)(lw * 5 + i) * 8 + (l >> 3);
    const unsigned R = min((unsigned)(mt * BM) + row, (unsigned)(M_ALL - 1));
    xlo[i] = (R >> 9) * (unsigned)SRC_B + (R & 511) * RB + ((((unsigned)l & 7) ^ ((row >> 1) & 7)) << 4);
  }
  for (int i = 0; i < 4; ++i) {
    const unsigned k = (unsigned)(lw * 4 + i) * 2 + (l >> 5);
    const unsigned cb = (l & 31) * 16;
    const unsigned sw = ((k & 3) << 6) | (((k >> 3) & 1) << 5) | (((k >> 2) & 1) << 4);
    wlo[i] = k * (N_LOC * 2) + (cb ^ sw);
  }
  #define GLDS(vo, sbase, ldsa, POL) asm volatile("s_mov_b32 m0, %2\n\ts_nop 0\n\tglobal_load_lds_dwordx4 %0, %1" POL :: "v"(vo), "s"(sbase), "s"(ldsa) : "memory")
  auto gl_w = [&](int slice, int slot) {
    const char* g = wbase + (size_t)slice * (BK * N_LOC * 2);
    for (int i = 0; i < 4; ++i) GLDS(wlo[i], g, (unsigned)(LDS_W0 + slot * WSL + (lw * 4 + i) * 1024), "");
  };
  auto gl_x = [&](int unit, int slot) {
    const char* g = xbase + (size_t)(unit / UPC) * CHUNK_B + (unit % UPC) * 128;
    for (int i = 0; i < 5; ++i) GLDS(xlo[i], g, (unsigned)(LDS_X0 + slot * XUN + (lw * 5 + i) * 1024), " sc0 sc1");
  };
  if (lw == 0) { wait_flags(flags, nflag, call, tmo); STAMP(1); }
  BAR();
  gl_w(0, 0); gl_x(0, 0); gl_w(1, 1); gl_w(2, 2); gl_x(1, 1);
  WAIT_V(13);
  BAR();
  int ws = 0;
  for (int p = 0; p < NSLICE; ++p) {
    if (p + 2 >= NSLICE) { WAIT_V(0); } else if (p & 1) { WAIT_V(4); } else { WAIT_V(9); }
    if (((p + 3) % SPC) == 0 && p + 3 < NSLICE && lw == 0) { wait_flags(flags + ((p + 3) / SPC) * 256, nflag, call, tmo); if (((p + 3) & 7) == 0) STAMP(1 + ((p + 3) >> 3)); }
    BAR();
    if (p + 3 < NSLICE) {
      gl_w(p + 3, ws);
      if (p & 1) gl_x((p + 3) >> 1, ((p + 3) >> 1) & 1);
    }
    ws = (ws == 2) ? 0 : ws + 1;
  }
  #undef GLDS
}

template <int GRP>
__device__ __forceinline__ void gemm_role(const Params& P, unsigned call, int gid, char* lds) {
  const int nt = gid & 7, mt = gid >> 3;
  const int tid = threadIdx.x, w = __builtin_amdgcn_readfirstlane(tid >> 6), l = tid & 63;
  constexpr int wm = GRP;
  const int wn = w & 3;
  unsigned* tmo = (unsigned*)(P.ws + WS_CTR + 16 * 128);

  unsigned xr0;
  {
    const unsigned r = wm * (BM / 2) + (l & 15);
    xr0 = LDS_X0 + r * 128 + ((((unsigned)(l >> 4)) ^ ((r >> 1) & 7)) << 4);
  }
  unsigned wr0;
  {
    const unsigned g = l >> 4, q = (l & 15) >> 2, p = l & 3;
    wr0 = LDS_W0 + (8 * g + q) * 512 + 8 * (p & 1) + ((wn * 128 + (p >> 1) * 16) ^ ((q << 6) | ((g & 1) << 5)));
  }
  f32x4 acc[4][MB];
  for (int a = 0; a < 4; ++a) for (int b = 0; b < MB; ++b) acc[a][b] = f32x4{0.f, 0.f, 0.f, 0.f};

  struct Frag { bf16x4 wl[4], wh[4]; bf16x8 x[MB]; };
  #define RD_W(f, nb, t, wb) do { unsigned _t; \
      asm volatile("v_xor_b32 %1, %3, %2\n\tds_read_b64_tr_b16 %0, %1 offset:%4" : "=v"((t) ? f.wh[nb] : f.wl[nb]), "=&v"(_t) \
                   : "v"(wb), "i"(((nb) << 5) | ((t) << 4)), "i"((t) * 2048)); } while (0)
  #define RD_X(f, mb, xo) asm volatile("ds_read_b128 %0, %1 offset:%2" : "=v"(f.x[mb]) : "v"(xo), "i"((mb) * 2048))
  auto pin_wait = [&](Frag& f) {
    asm volatile("s_waitcnt lgkmcnt(0)" : "+v"(f.wl[0]), "+v"(f.wl[1]), "+v"(f.wl[2]), "+v"(f.wl[3]),
                 "+v"(f.wh[0]), "+v"(f.wh[1]), "+v"(f.wh[2]), "+v"(f.wh[3]),
                 "+v"(f.x[0]), "+v"(f.x[1]), "+v"(f.x[2]), "+v"(f.x[3]), "+v"(f.x[4]) :: "memory");
  };
  #define G(nb, mb) acc[nb][mb] = __builtin_amdgcn_mfma_f32_16x16x32_bf16(__builtin_shufflevector(cur.wl[nb], cur.wh[nb], 0, 1, 2, 3, 4, 5, 6, 7), cur.x[mb], acc[nb][mb], 0, 0, 0)

  Frag fa, fb;
  BAR();
  BAR();
  for (int nb = 0; nb < 4; ++nb) { RD_W(fa, nb, 0, wr0); RD_W(fa, nb, 1, wr0); }
  for (int mb = 0; mb < MB; ++mb) RD_X(fa, mb, xr0);
  int ws = 0;

  auto body = [&](Frag& cur, Frag& nxt, int p, auto PARC, auto RDC) {
    constexpr int PAR = decltype(PARC)::value, R = decltype(RDC)::value;
    pin_wait(cur);
    SB(); BAR(); SB();
    const int wsn = (ws == 2) ? 0 : ws + 1;
    const unsigned so = wr0 + wsn * WSL;
    const unsigned xo = (xr0 ^ (unsigned)((PAR ^ 1) * 64)) + ((((p + 1) >> 1) & 1) * XUN);
    if (R) RD_W(nxt, 0, 0, so); SB(); G(0, 0); SB();
    if (R) RD_W(nxt, 0, 1, so); SB(); G(0, 1); SB();
    if (R) RD_W(nxt, 1, 0, so); SB(); G(0, 2); SB();
    if (R) RD_W(nxt, 1, 1, so); SB(); G(0, 3); SB();
    if (R) RD_W(nxt, 2, 0, so); SB(); G(0, 4); SB();
    if (R) RD_W(nxt, 2, 1, so); SB(); G(1, 0); SB();
    if (R) RD_W(nxt, 3, 0, so); SB(); G(1, 1); SB();
    if (R) RD_W(nxt, 3, 1, so); SB(); G(1, 2); SB();
    if (R) RD_X(nxt, 0, xo); SB(); G(1, 3); SB();
    if (R) RD_X(nxt, 1, xo); SB(); G(1, 4); SB();
    if (R) RD_X(nxt, 2, xo); SB(); G(2, 0); SB();
    if (R) RD_X(nxt, 3, xo); SB(); G(2, 1); SB();
    if (R) RD_X(nxt, 4, xo); SB(); G(2, 2); G(2, 3); G(2, 4); G(3, 0); G(3, 1); G(3, 2); G(3, 3); G(3, 4); SB();
    ws = wsn;
  };
  using I0 = std::integral_constant<int, 0>; using I1 = std::integral_constant<int, 1>;
  for (int p = 0; p < NSLICE - 2; p += 2) {
    body(fa, fb, p, I0{}, I1{});
    body(fb, fa, p + 1, I1{}, I1{});
  }
  body(fa, fb, NSLICE - 2, I0{}, I1{});
  body(fb, fa, NSLICE - 1, I1{}, I0{});
  if (w == 0) STAMP(17);
  {
    const bool bad = __hip_atomic_load(tmo, __ATOMIC_RELAXED, __HIP_MEMORY_SCOPE_AGENT) == call;
    char* ep = lds + w * (MB * 16 * 128);
    const int l = lane_now();
    const unsigned mrow = l & 15, g = l >> 4;
    for (int nb = 0; nb < 4; ++nb) for (int mb = 0; mb < MB; ++mb) {
      const f32x4 a = acc[nb][mb];
      u32x2 v; v.x = pack_bf16(silu_f(a[0]), silu_f(a[1])); v.y = pack_bf16(silu_f(a[2]), silu_f(a[3]));
      if (bad) { v.x = 0x7FC07FC0u; v.y = 0x7FC07FC0u; }
      const unsigned row = mb * 16 + mrow, gran = nb * 4 + g;
      *reinterpret_cast<u32x2*>(ep + row * 128 + ((gran ^ (row & 15)) << 3)) = v;
    }
    WAIT_L(0);
    const int row0 = mt * BM + wm * (BM / 2);
    unsigned short* obase = P.out + (size_t)row0 * N_LOC + nt * BN + wn * 64;
    for (int i = 0; i < 2 * MB; ++i) {
      const unsigned row = i * 8 + (l >> 3), c = l & 7, x = row & 15;
      u32x4 v = *reinterpret_cast<const u32x4*>(ep + row * 128 + ((c ^ (x >> 1)) << 4));
      if (x & 1) { u32x4 t; t.x = v.z; t.y = v.w; t.z = v.x; t.w = v.y; v = t; }
      if (row0 + (int)row < M_ALL) *reinterpret_cast<u32x4*>(obase + (size_t)row * N_LOC + c * 8) = v;
    }
  }
  if (w == 0) { WAIT_V(0); STAMP(18); }
  #undef G
  #undef RD_W
  #undef RD_X
}

__global__ void __launch_bounds__(NTHREADS, 3) fused_ag_gemm_silu(Params P) {
  extern __shared__ __attribute__((aligned(16))) char lds[];
  const int b = blockIdx.x;
  const int tid = threadIdx.x, wv = __builtin_amdgcn_readfirstlane(tid >> 6), l = tid & 63;
  const bool is_push = b < NPUSH;
  volatile unsigned* lseq = (volatile unsigned*)(lds + LDS_BYTES - 16);
  if (tid == 0) {
    unsigned* ctr = (unsigned*)(P.ws + WS_CTR + ((is_push ? 0 : 8) + (b & 7)) * 128);
    const unsigned t = __hip_atomic_fetch_add(ctr, 1u, __ATOMIC_RELAXED, __HIP_MEMORY_SCOPE_AGENT);
    *lseq = t / (unsigned)(is_push ? NPUSH / 8 : NGEMM / 8) + 1u;
  }
  __syncthreads();
  const unsigned call = __builtin_amdgcn_readfirstlane(*lseq);
  __syncthreads();
  if (wv == 0) STAMP(0);
  if (is_push) {
    if (wv < NR) pusher(P, call, b, wv, l);
  } else {
    const int gid = b - NPUSH;
    if (wv < 4) gemm_role<0>(P, call, gid, lds);
    else if (wv < 8) gemm_role<1>(P, call, gid, lds);
    else loader_role(P, call, gid, lds);
  }
}

}

extern "C" void kernel_launch(int rank, int nranks, ncclComm_t comm,
                              hipStream_t stream,
                              void* const* d_in, const int* in_sizes, int n_in,
                              void* d_out, int out_size,
                              const int* in_dtypes, int out_dtype,
                              void* const* d_out_all,
                              void* const* d_ws_all, size_t ws_bytes) {
  (void)comm; (void)in_dtypes; (void)out_dtype; (void)d_out_all;
  if (nranks != NR || n_in < 2 || in_sizes[0] != M_LOC * K_ALL || in_sizes[1] != K_ALL * N_LOC ||
      out_size != M_ALL * N_LOC || ws_bytes < WS_NEED) {
    fprintf(stderr, "kernel_launch: unexpected problem shape\n");
    return;
  }
  static bool attr_set = false;
  if (!attr_set) {
    HIP_CHECK(hipFuncSetAttribute((const void*)fused_ag_gemm_silu, hipFuncAttributeMaxDynamicSharedMemorySize, LDS_BYTES));
    attr_set = true;
  }
  Params P;
  P.x = (const unsigned short*)d_in[0];
  P.w = (const unsigned short*)d_in[1];
  P.out = (unsigned short*)d_out;
  P.ws = (char*)d_ws_all[rank];
  P.ws_all = d_ws_all;
  P.rank = rank;
  fused_ag_gemm_silu<<<GRID, NTHREADS, LDS_BYTES, stream>>>(P);
  HIP_CHECK(hipGetLastError());
}
```
